```python
import numpy as np

kernel_name = "gpu_a2a_gemm_m4096_k4096_n4096_e4m3e5m2_dqsilu"


ABSMAX_RTOL = 0.05

COMPUTE_DTYPE = "float32"
EPILOGUE = "dequant_silu"
IN_QUANT = None
OUT_QUANT = None
IN_FP8 = "e4m3e5m2"
IN_DTYPES = {"x": "e4m3", "w_mat": "e5m2",
             "scale_x": "float32", "scale_w": "float32"}
WORLD_SIZE = 8

def _snap_fp8(x, mant_bits, max_abs, min_normal):
    a = np.abs(x.astype(np.float64))
    _m, _e = np.frexp(a)
    q = np.where(a >= min_normal, np.ldexp(1.0, _e - 1 - mant_bits),
                 min_normal * 2.0 ** -mant_bits)
    snapped = np.minimum(np.round(a / q) * q, max_abs)
    return (np.sign(x) * snapped).astype(np.float32)

def setup_inputs():

    x = _snap_fp8(np.random.randn(4096, 4096).astype(np.float32), 3, 448.0, 2.0 ** -6)
    w_mat = _snap_fp8(np.random.randn(4096, 4096).astype(np.float32), 2, 57344.0, 2.0 ** -14)
    scale_x = (0.02 * (1.0 + np.random.rand(1))).astype(np.float32)
    scale_w = (0.02 * (1.0 + np.random.rand(1))).astype(np.float32)
    return {"x": x, "w_mat": w_mat, "scale_x": scale_x, "scale_w": scale_w}

def reference(x, w_mat, scale_x, scale_w):
    acc = (x @ w_mat).astype(np.float32)
    y = (acc * (scale_x * scale_w)).astype(np.float32)
    return (y / (1.0 + np.exp(-np.clip(y, -60.0, 60.0)))).astype(np.float32)
```

```cpp
#include <hip/hip_runtime.h>
#include <rccl/rccl.h>
#include <cstdio>
#include <cstdint>

#define HIP_CHECK(expr) do { hipError_t _e = (expr); if (_e != hipSuccess) { \
    fprintf(stderr, "HIP error %s at %s:%d\n", hipGetErrorString(_e), __FILE__, __LINE__); \
    return; } } while (0)

typedef int v8i __attribute__((ext_vector_type(8)));
typedef int v4i __attribute__((ext_vector_type(4)));
typedef int v2i __attribute__((ext_vector_type(2)));
typedef unsigned v4u __attribute__((ext_vector_type(4)));
typedef float v4f __attribute__((ext_vector_type(4)));
#define LDS_AS __attribute__((address_space(3)))

namespace {

constexpr int NRANKS = 8;
constexpr int NWG = 256;
constexpr int NTHREADS = 576;

constexpr unsigned WS_STAGE = 2u << 20;
constexpr unsigned WS_FLAGS = 4u << 20;
constexpr unsigned WS_CTR = WS_FLAGS + (64u << 10);
constexpr unsigned WS_TMO = WS_CTR + (64u << 10);
constexpr unsigned WS_DBG = WS_TMO + (64u << 10);
constexpr unsigned WS_NEEDED = WS_DBG + (256u << 10);

constexpr int NW = 6;
constexpr int NXS = 7;
constexpr int DW = 4;
constexpr unsigned LDS_X = NW * 16384;
constexpr unsigned LDS_CTRL = LDS_X + NXS * 8192;
constexpr unsigned LDS_BYTES = LDS_CTRL + 256;
constexpr int C_WDONE = 0;
constexpr int C_XDONE = 4;
constexpr int C_WRDY = 8;
constexpr int C_XRDY = 10;
constexpr int C_FAIL = 11;
constexpr int C_XAV = 12;

constexpr unsigned long long DEADLINE_TICKS = 100000000ull;

struct Args {
  const unsigned char* x;
  const unsigned char* w;
  const float* sx;
  const float* sw;
  float* out;
  unsigned char* ws[NRANKS];
  int rank;
  int nranks;
};

__device__ __forceinline__ v4i make_rsrc(const void* p, unsigned bytes) {
  unsigned long long a = (unsigned long long)p;
  v4i r;
  r[0] = __builtin_amdgcn_readfirstlane((int)(unsigned)a);
  r[1] = __builtin_amdgcn_readfirstlane((int)(unsigned)(a >> 32));
  r[2] = (int)bytes;
  r[3] = 0x00020000;
  return r;
}

__device__ __forceinline__ unsigned vm_outstanding() {
  unsigned v = __builtin_amdgcn_s_getreg((31 << 11) | (0 << 6) | 7);
  return (v & 15u) | ((v >> 18) & 48u);
}

__device__ __forceinline__ unsigned lds_ld(unsigned char LDS_AS* base, int word) {
  return __builtin_amdgcn_readfirstlane(*(volatile unsigned LDS_AS*)(base + LDS_CTRL + 4 * word));
}
__device__ __forceinline__ v4u lds_ld4(unsigned char LDS_AS* base, int word) {
  return *(volatile v4u LDS_AS*)(base + LDS_CTRL + 4 * word);
}
__device__ __forceinline__ void lds_st(unsigned char LDS_AS* base, int word, unsigned v) {
  *(volatile unsigned LDS_AS*)(base + LDS_CTRL + 4 * word) = v;
}
__device__ __forceinline__ unsigned umin4(v4u v) {
  unsigned a = v[0] < v[1] ? v[0] : v[1], b = v[2] < v[3] ? v[2] : v[3];
  return __builtin_amdgcn_readfirstlane(a < b ? a : b);
}

#define CBAR() asm volatile("" ::: "memory")

#define STAMP(i) do { } while (0)

#define GUARD(stmt) do { } while (0)

#define XAUX "sc1"

__device__ __forceinline__ unsigned unit_kbase(unsigned u, unsigned rank) {
  if (u < 8) return rank * 512 + u * 64;
  unsigned j = u - 8, f = (j * 37) >> 8, pi = j - 7 * f;
  return ((rank + 1 + pi) & 7) * 512 + f * 64;
}

__global__ void __launch_bounds__(NTHREADS) a2a_gemm_kernel(Args a) {
  extern __shared__ __attribute__((aligned(128))) unsigned char smem_raw[];
  unsigned char LDS_AS* smem = (unsigned char LDS_AS*)smem_raw;
  const int tid = threadIdx.x, lane = tid & 63;
  const int wv = __builtin_amdgcn_readfirstlane(tid >> 6);
  const unsigned b = blockIdx.x;
  const unsigned rank = (unsigned)a.rank;
  const unsigned q = b >> 3, mblk = q & 7, ntile = (b & 7) * 4 + (q >> 3);
  const unsigned lds0 = (unsigned)(uintptr_t)smem;

  if (wv == 8) {
    STAMP(0);
    const unsigned dest = ((b & 7) + q) & 7, piece = q;
    unsigned* ctr = (unsigned*)(a.ws[rank] + WS_CTR) + b * 2;
    unsigned c0 = __hip_atomic_load(ctr, __ATOMIC_RELAXED, __HIP_MEMORY_SCOPE_AGENT);
    const bool active = dest != rank && dest < (unsigned)a.nranks;
    v4u d[8];
    if (active) {
      __amdgpu_buffer_rsrc_t rx = __builtin_amdgcn_make_buffer_rsrc((void*)a.x, 0, 2u << 20, 0x00020000);
      const unsigned vo = (lane & 15) * 512 + (lane >> 4) * 16, so = (dest * 512 + piece * 16) * 512;
#pragma unroll
      for (int f = 0; f < 8; ++f) d[f] = __builtin_amdgcn_raw_buffer_load_b128(rx, vo + f * 64, so, 0);
    }
    __builtin_amdgcn_s_barrier();
    c0 = __builtin_amdgcn_readfirstlane(c0);
    if (lane == 0) __hip_atomic_store(ctr, c0 + 1, __ATOMIC_RELAXED, __HIP_MEMORY_SCOPE_AGENT);
    if (active) {
      unsigned char* wsd = a.ws[dest];
      __amdgpu_buffer_rsrc_t rd = __builtin_amdgcn_make_buffer_rsrc((void*)wsd, 0, WS_NEEDED, 0x00020000);
      const unsigned so = (c0 & 1) * WS_STAGE + ((rank * 8 + (piece >> 2)) * 32 + (piece & 3)) * 1024;
      const unsigned fo = WS_FLAGS + ((piece >> 2) * 64 + (piece & 3) * 16 + rank) * 4;
      const unsigned base = c0 * 8;
      const unsigned vo = lane * 16;
#define PUSH_S(f) do { __builtin_amdgcn_raw_buffer_store_b128(d[f], rd, vo, so + (f) * 4096, 17); CBAR(); } while (0)
#define PUSH_F(f) do { if (lane == 0) __builtin_amdgcn_raw_buffer_store_b32(base + (f) + 1, rd, 0, fo, 17); CBAR(); } while (0)
#define PUSH_W(n) asm volatile("s_waitcnt vmcnt(" #n ")" ::: "memory")
      PUSH_S(0); PUSH_S(1); PUSH_S(2);
      STAMP(1);
      PUSH_W(2); PUSH_F(0); PUSH_S(3);
      PUSH_W(3); PUSH_F(1); PUSH_S(4);
      PUSH_W(4); PUSH_F(2); PUSH_S(5);
      PUSH_W(4); PUSH_F(3); PUSH_S(6);
      PUSH_W(4); PUSH_F(4); PUSH_S(7);
      PUSH_W(4); PUSH_F(5);
      PUSH_W(3); PUSH_F(6);
      PUSH_W(2); PUSH_F(7);
      STAMP(2);
    }
    return;
  }

  if (wv == 7) {
    unsigned* ctr = (unsigned*)(a.ws[rank] + WS_CTR) + b * 2 + 1;
    unsigned c0 = __hip_atomic_load(ctr, __ATOMIC_RELAXED, __HIP_MEMORY_SCOPE_AGENT);
    if (lane < 16) lds_st(smem, lane, lane == C_XAV ? 8u : 0u);
    asm volatile("s_waitcnt lgkmcnt(0)" ::: "memory");
    __builtin_amdgcn_s_barrier();
    c0 = __builtin_amdgcn_readfirstlane(c0);
    const unsigned call = c0 + 1;
    if (lane == 0) __hip_atomic_store(ctr, call, __ATOMIC_RELAXED, __HIP_MEMORY_SCOPE_AGENT);
    const unsigned par8 = (c0 & 1) << 8;
    unsigned* tmo = (unsigned*)(a.ws[rank] + WS_TMO);
    const unsigned src = lane & 7, kind = (lane >> 3) & 1;
    const bool isflag = kind == 0 && src != rank;
    const unsigned pi = (src - rank - 1) & 7;
    const unsigned* p = (lane == 8) ? tmo : (const unsigned*)(a.ws[rank] + WS_FLAGS) + mblk * 64 + lane;
    const unsigned base = c0 * 8;
    const unsigned long long t0 = __builtin_amdgcn_s_memrealtime();
    unsigned R = 0;
    while (R < 56) {
      unsigned v = __hip_atomic_load(p, __ATOMIC_RELAXED, __HIP_MEMORY_SCOPE_SYSTEM);
      int prog = (int)(v - base);
      prog = prog < 0 ? 0 : (prog > 8 ? 8 : prog);
      unsigned val = isflag ? (unsigned)prog * 7 + pi : 63u;
      unsigned lo = 0;
#pragma unroll
      for (int bit = 5; bit >= 0; --bit) {
        unsigned t = lo + (1u << bit);
        if (__builtin_amdgcn_ballot_w64(val < t) == 0) lo = t;
      }
      if (lo > 56) lo = 56;
      if (lo > R) {
        R = lo;
        lds_st(smem, C_XAV, (8 + R) | par8);
      } else {
        bool dead = __builtin_amdgcn_readlane(v, 8) == call || __builtin_amdgcn_s_memrealtime() - t0 > DEADLINE_TICKS;
        if (dead) {
          if (lane == 0) __hip_atomic_store(tmo, call, __ATOMIC_RELAXED, __HIP_MEMORY_SCOPE_AGENT);
          lds_st(smem, C_FAIL, 1u);
          lds_st(smem, C_XAV, 64u | par8);
          break;
        }
        __builtin_amdgcn_s_sleep(1);
      }
    }
    return;
  }

  if (wv == 6) {
    const v4i rx = make_rsrc(a.x, 2u << 20);
    const v4i rs = make_rsrc(a.ws[rank], 2 * WS_STAGE);
    const unsigned vl0 = (lane & 15) * 512 + (lane >> 4) * 16;
    const unsigned vl1 = vl0 + 8192 - 1024, vl2 = vl0 + 2 * (8192 - 1024), vl3 = vl0 + 3 * (8192 - 1024);
    const unsigned vr = lane * 16;
    const unsigned xrow = (rank * 512 + mblk * 64) * 512;
#pragma unroll 1
    for (unsigned u = 0; u < 8; ++u) {
      unsigned m = lds0 + LDS_X + u * 4096, so = xrow + u * 64;
      asm volatile("s_mov_b32 m0, %0\n\ts_nop 0\n\t"
                   "buffer_load_dwordx4 %1, %5, %6 offen lds\n\t"
                   "buffer_load_dwordx4 %2, %5, %6 offen offset:1024 lds\n\t"
                   "buffer_load_dwordx4 %3, %5, %6 offen offset:2048 lds\n\t"
                   "buffer_load_dwordx4 %4, %5, %6 offen offset:3072 lds"
                   :: "s"(m), "v"(vl0), "v"(vl1), "v"(vl2), "v"(vl3), "s"(rx), "s"(so) : "memory");
    }
    __builtin_amdgcn_s_barrier();
    unsigned ni = 8, pub = 0, slot = 8, f = 0, pi = 0, lim = 2 * NXS, av = 8, par = 0;
    while (pub < 64) {
      if (ni < 64) {
        if (ni >= av) { unsigned v = lds_ld(smem, C_XAV); av = v & 255u; par = v >> 8; }
        if (ni >= lim) lim = 2 * umin4(lds_ld4(smem, C_XDONE)) + 2 * NXS;
        while (ni < av && ni < lim) {
          unsigned src = (rank + 1 + pi) & 7;
          unsigned m = lds0 + LDS_X + slot * 4096;
          unsigned so = par * WS_STAGE + ((src * 8 + mblk) * 8 + f) * 4096;
          asm volatile("s_mov_b32 m0, %0\n\ts_nop 0\n\t"
                       "buffer_load_dwordx4 %1, %2, %3 offen " XAUX " lds\n\t"
                       "buffer_load_dwordx4 %1, %2, %3 offen offset:1024 " XAUX " lds\n\t"
                       "buffer_load_dwordx4 %1, %2, %3 offen offset:2048 " XAUX " lds\n\t"
                       "buffer_load_dwordx4 %1, %2, %3 offen offset:3072 " XAUX " lds"
                       :: "s"(m), "v"(vr), "s"(rs), "s"(so) : "memory");
          ++ni;
          if (++slot == 2 * NXS) slot = 0;
          if (++pi == 7) { pi = 0; ++f; }
        }
      }
      unsigned landed = ni - ((vm_outstanding() + 3) >> 2);
      if (landed > pub) { pub = landed; lds_st(smem, C_XRDY, pub); }
      GUARD(return);
    }
    STAMP(3);
    return;
  }

  if (wv >= 4) {
    const unsigned h = wv - 4;
    const v4i rw = make_rsrc(a.w, 16u << 20);
    const unsigned sw0 = ((lane & 7) ^ ((lane >> 4) & 3)) * 16;
    const unsigned v0 = (lane >> 3) * 4096 + sw0;
    const unsigned v1 = v0 + 8 * 4096 - 1024;
    const unsigned v2 = (lane >> 3) * 4096 + (sw0 ^ 64) + 16 * 4096 - 2048;
    const unsigned v3 = v2 + 8 * 4096 - 1024;
    unsigned ti = 0, pub = 0, slot = 0, lim = NW;
    bool joined = false;
    while (pub < 32) {
      if (ti < 32) {
        if (ti >= lim) {
          if (!joined) { __builtin_amdgcn_s_barrier(); joined = true; }
          lim = umin4(lds_ld4(smem, C_WDONE)) + NW;
        }
        if (ti < lim) {
          unsigned m = lds0 + slot * 16384 + h * 8192;
          unsigned so = unit_kbase(2 * ti + h, rank) * 4096 + ntile * 128;
          asm volatile("s_mov_b32 m0, %0\n\ts_nop 0\n\t"
                       "buffer_load_dwordx4 %2, %6, %1 offen lds\n\t"
                       "buffer_load_dwordx4 %3, %6, %1 offen offset:1024 lds\n\t"
                       "buffer_load_dwordx4 %4, %6, %1 offen offset:2048 lds\n\t"
                       "buffer_load_dwordx4 %5, %6, %1 offen offset:3072 lds\n\t"
                       "s_add_u32 m0, m0, 4096\n\ts_add_u32 %1, %1, 131072\n\t"
                       "buffer_load_dwordx4 %2, %6, %1 offen lds\n\t"
                       "buffer_load_dwordx4 %3, %6, %1 offen offset:1024 lds\n\t"
                       "buffer_load_dwordx4 %4, %6, %1 offen offset:2048 lds\n\t"
                       "buffer_load_dwordx4 %5, %6, %1 offen offset:3072 lds"
                       : "+s"(m), "+s"(so) : "v"(v0), "v"(v1), "v"(v2), "v"(v3), "s"(rw) : "memory", "scc");
          ++ti;
          if (++slot == NW) slot = 0;
          if (!joined) continue;
        }
      }
      unsigned landed = ti - ((vm_outstanding() + 7) >> 3);
      if (landed > pub) { pub = landed; lds_st(smem, C_WRDY + h, pub); }
      GUARD(return);
    }
    STAMP(4 + h);
    return;
  }

  {
    const float scale = a.sx[0] * a.sw[0];
    const unsigned g = lane >> 4, qq = (lane & 15) >> 1, pp = lane & 1;
    const unsigned F = ((qq >> 1) & 3) | ((g & 1) << 2);
    const unsigned wa0 = 128 * (16 * g + qq) + 8 * pp + 16 * ((2 * wv) ^ F);
    const unsigned xa0 = LDS_X + lane * 16;
    v4f acc[4][2];
#pragma unroll
    for (int i = 0; i < 4; ++i) { acc[i][0] = (v4f){0, 0, 0, 0}; acc[i][1] = (v4f){0, 0, 0, 0}; }
    v8i wf[DW][2];
    unsigned wrdy = 0, xrdy = 0, wslot = 0, xslot = 0;
    __builtin_amdgcn_s_barrier();

    auto read_w = [&](unsigned t, v8i* dst) {
      while (wrdy <= t) {
        v4u c = lds_ld4(smem, C_WRDY);
        unsigned m = c[0] < c[1] ? c[0] : c[1];
        wrdy = __builtin_amdgcn_readfirstlane(m);
        xrdy = __builtin_amdgcn_readfirstlane(c[2]);
        GUARD(wrdy = 99);
      }
      CBAR();
      unsigned char LDS_AS* p0 = smem + wslot * 16384 + wa0;
      unsigned char LDS_AS* p1 = smem + wslot * 16384 + (wa0 ^ 16);
#pragma unroll
      for (int t4 = 0; t4 < 4; ++t4) {
        const unsigned o = (t4 >> 1) * 8192 + (t4 & 1) * 1024;
        v2i r0 = __builtin_amdgcn_ds_read_tr8_b64_v2i32((v2i LDS_AS*)(p0 + o));
        v2i r1 = __builtin_amdgcn_ds_read_tr8_b64_v2i32((v2i LDS_AS*)(p1 + o));
        dst[0][2 * t4] = r0[0]; dst[0][2 * t4 + 1] = r0[1];
        dst[1][2 * t4] = r1[0]; dst[1][2 * t4 + 1] = r1[1];
      }
      CBAR();
      lds_st(smem, C_WDONE + wv, t + 1);
      if (++wslot == NW) wslot = 0;
    };

#pragma unroll
    for (int d = 0; d < DW; ++d) read_w(d, wf[d]);

#pragma unroll 1
    for (unsigned tt = 0; tt < 32; tt += DW) {
#pragma unroll
      for (int d = 0; d < DW; ++d) {
        const unsigned t = tt + d;
        while (xrdy < 2 * t + 2) {
          v4u c = lds_ld4(smem, C_WRDY);
          unsigned m = c[0] < c[1] ? c[0] : c[1];
          wrdy = __builtin_amdgcn_readfirstlane(m);
          xrdy = __builtin_amdgcn_readfirstlane(c[2]);
          GUARD(xrdy = 99);
        }
        CBAR();
        unsigned char LDS_AS* px = smem + xa0 + xslot * 8192;
        v8i xf[4];
#pragma unroll
        for (int mt = 0; mt < 4; ++mt) {
          v4i lo = *(v4i LDS_AS*)(px + mt * 1024);
          v4i hi = *(v4i LDS_AS*)(px + 4096 + mt * 1024);
          xf[mt] = (v8i){lo[0], lo[1], lo[2], lo[3], hi[0], hi[1], hi[2], hi[3]};
        }
        CBAR();
        lds_st(smem, C_XDONE + wv, t + 1);
        if (++xslot == NXS) xslot = 0;
#pragma unroll
        for (int mt = 0; mt < 4; ++mt) {
          acc[mt][0] = __builtin_amdgcn_mfma_scale_f32_16x16x128_f8f6f4(wf[d][0], xf[mt], acc[mt][0], 1, 0, 0, 0, 0, 0);
          acc[mt][1] = __builtin_amdgcn_mfma_scale_f32_16x16x128_f8f6f4(wf[d][1], xf[mt], acc[mt][1], 1, 0, 0, 0, 0, 0);
        }
        if (t + DW < 32) read_w(t + DW, wf[d]);
      }
    }
    if (wv == 0) STAMP(6);
    const bool fail = lds_ld(smem, C_FAIL) != 0;
    float* o = a.out + (size_t)(mblk * 64 + (lane & 15)) * 4096 + ntile * 128 + wv * 32 + g * 4;
#pragma unroll
    for (int mt = 0; mt < 4; ++mt) {
#pragma unroll
      for (int c2 = 0; c2 < 2; ++c2) {
        v4f r;
#pragma unroll
        for (int e = 0; e < 4; ++e) {
          float y = acc[mt][c2][e] * scale;
          float yc = __builtin_amdgcn_fmed3f(y, -60.0f, 60.0f);
          float ex = __builtin_amdgcn_exp2f(yc * -1.4426950408889634f);
          r[e] = y * __builtin_amdgcn_rcpf(1.0f + ex);
        }
        if (fail) r = (v4f){__builtin_nanf(""), __builtin_nanf(""), __builtin_nanf(""), __builtin_nanf("")};
        *(v4f*)(o + (size_t)mt * 16 * 4096 + c2 * 16) = r;
      }
    }
    if (wv == 0) STAMP(7);
  }
}

}

extern "C" void kernel_launch(int rank, int nranks, ncclComm_t comm, hipStream_t stream,
                              void* const* d_in, const int* in_sizes, int n_in,
                              void* d_out, int out_size,
                              const int* in_dtypes, int out_dtype,
                              void* const* d_out_all,
                              void* const* d_ws_all, size_t ws_bytes) {
  (void)comm; (void)in_sizes; (void)in_dtypes; (void)out_dtype; (void)d_out_all;
  if (nranks != NRANKS || (unsigned)rank >= (unsigned)NRANKS || n_in < 4 || ws_bytes < WS_NEEDED ||
      out_size != 512 * 4096) {
    fprintf(stderr, "a2a_gemm: unsupported configuration\n");
    return;
  }
  HIP_CHECK(hipFuncSetAttribute((const void*)a2a_gemm_kernel, hipFuncAttributeMaxDynamicSharedMemorySize, LDS_BYTES));
  Args a;
  a.x = (const unsigned char*)d_in[0];
  a.w = (const unsigned char*)d_in[1];
  a.sx = (const float*)d_in[2];
  a.sw = (const float*)d_in[3];
  a.out = (float*)d_out;
  for (int p = 0; p < NRANKS; ++p) a.ws[p] = (unsigned char*)d_ws_all[p];
  a.rank = rank;
  a.nranks = nranks;
  hipLaunchKernelGGL(a2a_gemm_kernel, dim3(NWG), dim3(NTHREADS), LDS_BYTES, stream, a);
  HIP_CHECK(hipGetLastError());
}
```

```cpp
#include <hip/hip_runtime.h>
#include <rccl/rccl.h>
#include <cstdio>
#include <cstdint>

#define HIP_CHECK(expr) do { hipError_t _e = (expr); if (_e != hipSuccess) { \
    fprintf(stderr, "HIP error %s at %s:%d\n", hipGetErrorString(_e), __FILE__, __LINE__); \
    return; } } while (0)

typedef int v16i __attribute__((ext_vector_type(16)));
typedef int v4i __attribute__((ext_vector_type(4)));
typedef int v2i __attribute__((ext_vector_type(2)));
typedef unsigned v4u __attribute__((ext_vector_type(4)));
typedef unsigned v2u __attribute__((ext_vector_type(2)));
typedef float v4f __attribute__((ext_vector_type(4)));
#define LDS_AS __attribute__((address_space(3)))

namespace {

constexpr int NRANKS = 8;
constexpr int NWG = 256;
#define PF2 1
constexpr int NTHREADS = 640;

constexpr unsigned WS_STAGE = 2u << 20;
constexpr unsigned WS_FLAGS = 6u << 20;
constexpr unsigned WS_CTR = WS_FLAGS + (64u << 10);
constexpr unsigned WS_DBG = WS_CTR + (64u << 10);
constexpr unsigned WS_NEEDED = WS_DBG + (256u << 10);
#define LAYOUT2 1
#define LAYOUT3 1

#define NW_ 6
#define NXS_ 7
constexpr int NW = NW_;
constexpr int NXS = NXS_;
constexpr unsigned LDS_X = NW * 16384;
constexpr unsigned LDS_CTRL = LDS_X + NXS * 8192;
constexpr unsigned LDS_BYTES = LDS_CTRL + 256 + 1024;
constexpr int C_DONE = 0;
constexpr int C_RETRY = 1;
constexpr int C_XAV = 2;
constexpr int C_GO = 3;
constexpr int C_TAB = 32;
constexpr int C_FLG = 64;
constexpr int C_FAIL = C_FLG + 4 * 32 + 3;
constexpr int C_XFIX = C_FAIL + 1;
constexpr unsigned OOB = 0x40000000u;

constexpr unsigned long long DEADLINE_TICKS = 100000000ull;

struct alignas(64) Args {
  const unsigned char* x;
  const unsigned char* w;
  const float* sx;
  const float* sw;
  float* out;
  unsigned char* ws_self;
  int rank;
  int nranks;
  int pad[2];
  unsigned char* ws[NRANKS];
  unsigned char* wn[NRANKS];
};

__device__ __forceinline__ v4i make_rsrc(int lo, int hi, unsigned bytes) {
  v4i r;
  r[0] = lo;
  r[1] = hi & 0xffff;
  r[2] = (int)bytes;
  r[3] = 0x00020000;
  return r;
}
__device__ __forceinline__ unsigned long long mk64(int lo, int hi) {
  return (unsigned long long)(unsigned)lo | ((unsigned long long)(unsigned)hi << 32);
}
__device__ __forceinline__ void lds_st1(unsigned addr, unsigned v) {
  asm volatile("s_mov_b64 exec, 1\n\tds_write_b32 %0, %1\n\ts_mov_b64 exec, -1" :: "v"(addr), "v"(v) : "memory");
}
__device__ __forceinline__ unsigned next_ctr(unsigned c) { return ((c + 4) & ~3u) | ((c & 3) == 2 ? 0u : (c & 3) + 1); }
__device__ __forceinline__ unsigned now32() { return (unsigned)__builtin_amdgcn_s_memrealtime(); }

#define STAMPV(i, v) do { } while (0)
#define TS(name) do { } while (0)
#define STAMP(i) do { } while (0)

#define PAUX "sc0 sc1"
#define ZAUX "sc0 sc1"
#define OAUX 0
#define XAUX "sc1"
#define SLOT4 62
#define D0 110
#define SLACK 30
#define MG 10
#define LA 80
#define ADV 0
#define QD 6

__global__ void __launch_bounds__(NTHREADS) a2a_gemm_kernel(Args a_unused) {
  extern __shared__ __attribute__((aligned(128))) unsigned char smem_raw[];
  unsigned char LDS_AS* smem = (unsigned char LDS_AS*)smem_raw;
  const unsigned b = blockIdx.x;
  const unsigned long long ts0 = __builtin_amdgcn_s_memrealtime();
  const int tid = threadIdx.x, lane = tid & 63;
  const int wv = __builtin_amdgcn_readfirstlane(tid >> 6);
  const unsigned lds0 = (unsigned)(uintptr_t)smem;
  if (wv == 4) {
    const unsigned iv = lane == C_XAV ? 4u : 0u;
    const unsigned fv = lane <= 32 ? 1u : 0u;
    const v4u f4 = {fv, fv, fv, 0u};
    asm volatile("ds_write_b32 %0, %1\n\tds_write_b128 %2, %3\n\ts_waitcnt lgkmcnt(0)"
                 :: "v"(lds0 + LDS_CTRL + 4 * lane), "v"(iv), "v"(lds0 + LDS_CTRL + 4 * C_FLG + 16 * lane), "v"(f4) : "memory");
  }
  v16i ka;
  v2i wsd, wsn;
  const unsigned q7 = (b >> 3) * 37 >> 8, r7 = (b >> 3) - 7 * q7;
  asm volatile("s_load_dwordx16 %0, %3, 0x0\n\ts_load_dwordx2 %1, %3, %4 offset:0x40\n\t"
               "s_load_dwordx2 %2, %3, %5 offset:0x80\n\ts_barrier"
               : "=&s"(ka), "=&s"(wsd), "=&s"(wsn)
               : "s"(__builtin_amdgcn_kernarg_segment_ptr()), "s"((b & 7) * 8), "s"(r7 * 8) : "memory");
#define NTM 0
  const unsigned q = b >> 3, mblk = q & 7;
  const unsigned ntile = NTM == 0 ? (b & 7) * 4 + (q >> 3) : NTM == 1 ? (q >> 3) * 8 + (b & 7) : ((b & 7) * 2 + ((q >> 3) & 1)) + (q >> 4) * 16;
#define HS 0
  const unsigned vx = (lane >> 3) * 512 + ((lane & 7) ^ (lane >> 4)) * 16, vy = (vx ^ 64) + 4096;
  const unsigned vl = lane * 16;
#define OROT 2
  const unsigned r0 = OROT == 0 ? 0 : OROT == 1 ? mblk & 3 : OROT == 2 ? (mblk + (q >> 3)) & 3 : ((mblk >> 1) + (q >> 3)) & 3;
  asm volatile("s_waitcnt lgkmcnt(0)" : "+s"(ka), "+s"(wsd), "+s"(wsn));
  const unsigned rank = (unsigned)ka[12];
  const bool split = HS == 2 || (HS == 1 && ((ntile + ((unsigned)ka[2] >> 7)) & 3) == 3);
  const unsigned cb = split ? (ntile & ~4u) * 128 + (ntile & 4) * 16 : ntile * 128, hx = split ? 448 : 0;
  const v4i rs = make_rsrc(ka[10], ka[11], WS_NEEDED);
  const bool own = (b & 7) == rank;
  const unsigned dest = own ? (rank + 1 + r7) & 7 : b & 7, qs = q;
  const bool active = q < 28 && dest < (unsigned)ka[13];
#define GATE() do { if (active && qs < QD) { \
    while (*(volatile unsigned LDS_AS*)(smem + LDS_CTRL + 4 * C_GO) == 0) __builtin_amdgcn_s_sleep(1); } } while (0)

#define MONS 4u
#define PF_T0 80
#define PF_DT 60
#define PF3_T0 80
#define PF3_DT 12
#define PF3_R 4
#define PF5 20
#define PF5P 0
#define PF5T 130
#define APF_A 2
#define APF_J 0
#define PF_J0 1
  if (wv == 9) {
    const unsigned cl = (ntile + ((unsigned)ka[2] >> 7)) & 31;
    {
      const unsigned g = (3 - cl + (q >> 3)) & 3, cs = (cl - (q >> 3) + g) & 31;
      const bool sl = (cl & 3) == 3;
      unsigned d = 0, e = 0;
      const bool skip = ((cs & 7) != 3 && cs != 31) || (sl && mblk < 4 && !PF5P);
      const v4i rw = make_rsrc(ka[2], ka[3], 16u << 20);
      const unsigned vo = lane * 4096 + ((b & 7) * 4 + g) * 128;
      const unsigned u0 = !sl ? PF5 + 4 * PF5P + 23 - ((cl & 3) * 8 + mblk) : mblk < 4 ? PF5 + mblk : mblk - 4;
      const unsigned ue = skip ? 0 : !sl ? 42 : mblk < 4 ? PF5 + 4 * PF5P : PF5;
#pragma unroll 1
      for (unsigned u = u0, k = 0; u < ue; u += sl ? 4 : 24, ++k) {
        const unsigned j = u / 14, r = u - j * 14;
        const unsigned so = ((((rank + 1 + (r >> 1)) & 7) * 4 + j + 1) * 2 + (r & 1)) << 18;
        while ((int)(now32() - ((unsigned)ts0 + (sl ? PF_T0 : PF5T) + k * (PF_DT / 2))) < 0) __builtin_amdgcn_s_sleep(2);
        asm volatile("buffer_load_dword %0, %1, %2, %3 offen" : "+v"(d) : "v"(vo), "s"(rw), "s"(so) : "memory");
      }
      asm volatile("s_waitcnt vmcnt(0)" : "+v"(d), "+v"(e) :: "memory");
      STAMP(22);
      return;
    }
    if (mblk == 7 || (PF2 == 1 && (cl & 7) != 3 && cl != 31) || (PF2 == 3 && (cl & 3) != 3)) return;
    const v4i rw = make_rsrc(ka[2], ka[3], 16u << 20);
    const unsigned vo = lane * 4096 + ntile * 128, v2 = vo + 64 * 4096;
#pragma unroll 1
    for (unsigned j = PF_J0; j < 4; ++j) {
      const unsigned so = (((rank + 1 + mblk) & 7) * 4 + j) << 19;
      unsigned d, e;
      while ((int)(now32() - ((unsigned)ts0 + PF_T0 + (j - PF_J0) * PF_DT)) < 0) __builtin_amdgcn_s_sleep(4);
      asm volatile("buffer_load_dword %0, %2, %4, %5 offen\n\tbuffer_load_dword %1, %3, %4, %5 offen"
                   : "=&v"(d), "=&v"(e) : "v"(vo), "v"(v2), "s"(rw), "s"(so) : "memory");
    }
    asm volatile("s_waitcnt vmcnt(0)" ::: "memory");
    STAMP(22);
    return;
  }
  if (wv == 0) {
    TS(ts1);
    __builtin_amdgcn_s_setprio(3);
    const unsigned cto = WS_CTR + b * 128;
    unsigned c0;
    if (active) {
      const unsigned jj = q7, m = (r7 + (!own && r7 >= ((dest - rank - 1) & 7)) + dest - rank) & 7;
      const v4i rx = make_rsrc(ka[0], ka[1], 2u << 20);
      const v4i rd = make_rsrc(own ? wsn[0] : wsd[0], own ? wsn[1] : wsd[1], WS_NEEDED);
      const unsigned xo = (dest * 512 + m * 64) * 512 + jj * 128;
      const unsigned fo = WS_FLAGS + m * 2048 + ((((rank - dest) & 7) + m + 6) % 7) * 4 + 4;
      const unsigned slot = (unsigned)ts0 + D0 + ((qs * SLOT4) >> 2) - (qs < QD ? 0 : ADV);
      v4u d0, d1, d2, d3, d4, d5, d6, d7;
      unsigned vc, t;
      unsigned long long tn;
      asm volatile("s_buffer_load_dword %[c0], %[rs], %[cto] glc\n\t"
                   "buffer_load_dwordx4 %[d0], %[vx], %[rx], %[x0] offen\n\t"
                   "buffer_load_dwordx4 %[d1], %[vy], %[rx], %[x0] offen\n\t"
                   "buffer_load_dwordx4 %[d2], %[vx], %[rx], %[x1] offen\n\t"
                   "buffer_load_dwordx4 %[d3], %[vy], %[rx], %[x1] offen\n\t"
                   "buffer_load_dwordx4 %[d4], %[vx], %[rx], %[x2] offen\n\t"
                   "buffer_load_dwordx4 %[d5], %[vy], %[rx], %[x2] offen\n\t"
                   "buffer_load_dwordx4 %[d6], %[vx], %[rx], %[x3] offen\n\t"
                   "buffer_load_dwordx4 %[d7], %[vy], %[rx], %[x3] offen\n\t"
                   "s_cmp_gt_u32 %[q], 7\n\t"
                   "s_waitcnt lgkmcnt(0)\n\t"
                   "s_cbranch_scc1 Lnh%=\n\t"
                   "s_memrealtime %[tn]\n\t"
                   "s_lshr_b32 %[t], %[c0], 2\n\ts_add_u32 %[t], %[t], 1\n\ts_lshl_b32 %[t], %[t], 11\n\t"
                   "s_mov_b64 exec, 0xff\n\t"
                   "s_waitcnt lgkmcnt(0)\n\t"
                   "s_sub_u32 s90, %[due], s90\n\ts_max_i32 s90, s90, 0\n\ts_min_i32 s90, s90, 0xff\n\t"
                   "s_or_b32 %[t], %[t], s90\n\tv_mov_b32 %[vc], %[t]\n\t"
                   "buffer_store_dword %[vc], %[vf], %[rd], %[fo] offen " PAUX "\n\t"
                   "s_mov_b64 exec, -1\n\t"
                   "s_waitcnt vmcnt(1)\n\t"
                   "s_branch Lh%=\n\t"
                   "Lnh%=:\n\t"
                   "s_waitcnt vmcnt(0)\n\t"
                   "Lh%=:"
                   : [d0] "=&v"(d0), [d1] "=&v"(d1), [d2] "=&v"(d2), [d3] "=&v"(d3), [d4] "=&v"(d4), [d5] "=&v"(d5),
                     [d6] "=&v"(d6), [d7] "=&v"(d7), [vc] "=&v"(vc), [c0] "=&s"(c0), [t] "=&s"(t),
                     [tn] "=&{s[90:91]}"(tn)
                   : [vx] "v"(vx), [vy] "v"(vy), [rx] "s"(rx), [x0] "s"(xo), [x1] "s"(xo + 8192u), [x2] "s"(xo + 16384u),
                     [x3] "s"(xo + 24576u), [rs] "s"(rs), [cto] "s"(cto), [q] "s"(qs + 1), [due] "s"(slot + SLOT4 / 4),
                     [rd] "s"(rd), [fo] "s"(fo), [vf] "v"(vl * 16)
                   : "memory", "scc");
      TS(ts2);
      c0 = __builtin_amdgcn_readfirstlane(c0);
      const unsigned so = (c0 & 3) * WS_STAGE + m * 262144 + (rank * 4 + jj) * 8192;
      d0[0] = d0[0] ? d0[0] : 0x80u; d1[0] = d1[0] ? d1[0] : 0x80u; d2[0] = d2[0] ? d2[0] : 0x80u;
      d3[0] = d3[0] ? d3[0] : 0x80u; d4[0] = d4[0] ? d4[0] : 0x80u; d5[0] = d5[0] ? d5[0] : 0x80u;
      d6[0] = d6[0] ? d6[0] : 0x80u; d7[0] = d7[0] ? d7[0] : 0x80u;
      int left;
      while ((left = (int)(slot - now32())) > 0) { if (left > 30) __builtin_amdgcn_s_sleep(4); }
      int lead = left + 2 * SLOT4;
      lead = lead < 0 ? 0 : lead;
      const unsigned fv = (((c0 >> 2) + 1) << 11) | ((jj + 1) << 8) | (unsigned)lead;
      asm volatile("buffer_store_dwordx4 %0, %8, %9, %10 offen " PAUX "\n\t"
                   "buffer_store_dwordx4 %1, %8, %9, %10 offen offset:1024 " PAUX "\n\t"
                   "buffer_store_dwordx4 %2, %8, %9, %10 offen offset:2048 " PAUX "\n\t"
                   "buffer_store_dwordx4 %3, %8, %9, %10 offen offset:3072 " PAUX "\n\t"
                   "buffer_store_dwordx4 %4, %8, %9, %11 offen " PAUX "\n\t"
                   "buffer_store_dwordx4 %5, %8, %9, %11 offen offset:1024 " PAUX "\n\t"
                   "buffer_store_dwordx4 %6, %8, %9, %11 offen offset:2048 " PAUX "\n\t"
                   "buffer_store_dwordx4 %7, %8, %9, %11 offen offset:3072 " PAUX "\n\t"
                   "s_mov_b64 exec, 0xff\n\tbuffer_store_dword %12, %15, %9, %13 offen " PAUX "\n\t"
                   "ds_write_b32 %14, %12\n\ts_mov_b64 exec, -1"
                   :: "v"(d0), "v"(d1), "v"(d2), "v"(d3), "v"(d4), "v"(d5), "v"(d6), "v"(d7), "v"(vl), "s"(rd),
                      "s"(so), "s"(so + 4096u), "v"(fv), "s"(fo), "v"(lds0 + LDS_CTRL + 4 * C_GO), "v"(vl * 16)
                   : "memory");
    } else {
      asm volatile("s_buffer_load_dword %0, %1, %2 glc\n\ts_waitcnt lgkmcnt(0)" : "=s"(c0) : "s"(rs), "s"(cto) : "memory");
    }
    if (((b >> 2) & 7) != rank) {
      const unsigned zb = ((c0 & 3) == 0 ? 2u : (c0 & 3) - 1) * WS_STAGE + b * 8192;
      const v4u z = {0, 0, 0, 0};
      asm volatile("buffer_store_dwordx4 %0, %1, %2, %3 offen " ZAUX "\n\t"
                   "buffer_store_dwordx4 %0, %1, %2, %3 offen offset:1024 " ZAUX "\n\t"
                   "buffer_store_dwordx4 %0, %1, %2, %3 offen offset:2048 " ZAUX "\n\t"
                   "buffer_store_dwordx4 %0, %1, %2, %3 offen offset:3072 " ZAUX "\n\t"
                   "buffer_store_dwordx4 %0, %1, %2, %4 offen " ZAUX "\n\t"
                   "buffer_store_dwordx4 %0, %1, %2, %4 offen offset:1024 " ZAUX "\n\t"
                   "buffer_store_dwordx4 %0, %1, %2, %4 offen offset:2048 " ZAUX "\n\t"
                   "buffer_store_dwordx4 %0, %1, %2, %4 offen offset:3072 " ZAUX
                   :: "v"(z), "v"(vl), "s"(rs), "s"(zb), "s"(zb + 4096u) : "memory");
    }
    asm volatile("s_mov_b64 exec, 1\n\tbuffer_store_dword %0, off, %1, %2 sc1\n\ts_mov_b64 exec, -1\n\ts_waitcnt vmcnt(0)"
                 :: "v"(next_ctr(c0)), "s"(rs), "s"(cto) : "memory");
    STAMPV(0, ts0); STAMPV(18, ts1); STAMP(19);
    return;
  }

  if (wv == 8) {
    __builtin_amdgcn_s_setprio(3);
    const unsigned vi = (lane >> 3) & 3;
    const unsigned vid = ((rank + 1 + ((lane & 7) + 13 - mblk) % 7) & 7) * 4 + vi + 1;
    unsigned long long ta, tb;
    unsigned sch, arr, ann, tmp, thr, nr, xav, ns, t, u;
    unsigned vt = 0, ve = 255, vlast = ~0u;
#define SC_NP
#define SC_TR
#define SC_OUT
#define SC_INIT "s_mov_b32 %[arr], 0\n\t"
#define SC_BC(n, s) "s_mov_b32 exec_lo, 0x01010101 << " #n "\n\tv_mov_b32 v20, " s "\n\t"
    asm volatile("s_buffer_load_dword %[thr], %[rs], %[cto] glc\n\t"
                 "s_buffer_load_dwordx8 s[80:87], %[rs], %[fp] glc\n\t"
                 "s_memrealtime s[90:91]\n\t"
                 "s_mov_b32 exec_hi, 0\n\t"
                 "s_mov_b64 s[72:73], 0\n\ts_mov_b64 s[74:75], 0\n\ts_mov_b64 s[76:77], 0\n\ts_mov_b64 s[78:79], 0\n\t"
                 "s_mov_b32 %[sch], 0\n\t" SC_INIT "s_mov_b32 %[ann], 0\n\ts_mov_b32 %[nr], 0\n\t"
                 "s_mov_b32 %[xav], 4\n\ts_mov_b32 %[ns], 4\n\t"
                 "s_waitcnt lgkmcnt(0)\n\t"
                 "s_lshr_b32 %[thr], %[thr], 2\n\ts_add_u32 %[thr], %[thr], 1\n\ts_lshl_b32 %[thr], %[thr], 11\n\t"
                 "Ls%=:\n\t"
                 "s_memrealtime s[90:91]\n\t"
                 "s_cmp_eq_u32 s80, 1\n\ts_cbranch_scc1 Lt%=\n\t"
                 "s_xor_b64 %[ta], s[80:81], s[72:73]\n\ts_xor_b64 %[tb], s[82:83], s[74:75]\n\ts_or_b64 %[ta], %[ta], %[tb]\n\t"
                 "s_xor_b64 %[tb], s[84:85], s[76:77]\n\ts_or_b64 %[ta], %[ta], %[tb]\n\t"
                 "s_xor_b64 %[tb], s[86:87], s[78:79]\n\ts_or_b64 %[ta], %[ta], %[tb]\n\t"
                 "s_mov_b64 s[72:73], s[80:81]\n\ts_mov_b64 s[74:75], s[82:83]\n\t"
                 "s_mov_b64 s[76:77], s[84:85]\n\ts_mov_b64 s[78:79], s[86:87]\n\t"
                 "s_mov_b32 s80, 1\n\t"
                 "s_buffer_load_dwordx8 s[80:87], %[rs], %[fp] glc\n\t"
                 "s_cbranch_scc0 Ln%=\n\t"
                 SC_BC(1, "s73") SC_BC(2, "s74") SC_BC(3, "s75") SC_BC(4, "s76") SC_BC(5, "s77") SC_BC(6, "s78") SC_BC(7, "s79")
                 "s_mov_b32 exec_lo, 0xfefefefe\n\t"
                 "v_subrev_u32 v21, %[thr], v20\n\t"
                 "v_and_b32 v24, 0xff, v20\n\t"
                 "v_ashrrev_i32 v21, 8, v21\n\t"
                 "v_add_u32 v24, s90, v24\n\t"
                 "v_cmp_gt_i32 vcc, v21, %[vlast]\n\t"
                 "v_sub_u32 v23, %[vi], v21\n\t"
                 "v_max_i32 %[vlast], v21, %[vlast]\n\t"
                 "v_mul_i32_i24 v23, %[per], v23\n\t"
                 "s_or_b32 %[ann], %[ann], vcc_lo\n\t"
                 "v_add3_u32 v23, v23, v24, %[mg]\n\t"
                 "v_cndmask_b32 %[vt], %[vt], v23, vcc\n\t"
                 "v_cmp_gt_i32 vcc, %[vlast], %[vi]\n\t"
                 "s_mov_b32 exec_lo, -1\n\t"
                 "s_or_b32 %[t], %[thr], 0x80\n\t"
                 "s_or_b32 %[arr], %[arr], vcc_lo\n\t"
                 "s_cmp_eq_u32 s72, %[t]\n\ts_cbranch_scc1 Lf%=\n\t"
                 "Ln%=:\n\t"
                 SC_NP
                 "Lt%=:\n\t"
                 "v_mov_b32 v22, s90\n\t"
                 "v_sub_u32 v21, %[vt], v22\n\t"
                 "v_cmp_ge_i32 vcc, %[la], v21\n\t"
                 "s_and_b32 %[t], vcc_lo, %[ann]\n\t"
                 "v_cmp_ge_i32 vcc, 0, v21\n\t"
                 "s_andn2_b32 %[t], %[t], %[sch]\n\t"
                 "s_andn2_b32 %[tmp], %[arr], %[sch]\n\t"
                 "s_cselect_b32 %[tmp], %[tmp], %[t]\n\t"
                 "s_and_b32 %[u], vcc_lo, %[ann]\n\ts_or_b32 %[u], %[u], %[arr]\n\t"
                 "s_cmp_eq_u32 %[tmp], 0\n\ts_cbranch_scc1 Lr%=\n\t"
                 "s_mov_b32 exec_lo, %[tmp]\n\t"
                 "v_mbcnt_lo_u32_b32 v21, %[tmp], 0\n\t"
                 "s_or_b32 %[sch], %[sch], %[tmp]\n\t"
                 "v_add_u32 %[ve], %[ns], v21\n\t"
                 "s_bcnt1_i32_b32 %[tmp], %[tmp]\n\t"
                 "v_lshl_add_u32 v21, %[ve], 2, %[tab]\n\t"
                 "s_add_u32 %[ns], %[ns], %[tmp]\n\t"
                 "ds_write_b32 v21, %[vid]\n\t"
                 "s_mov_b32 exec_lo, -1\n\t"
                 "s_mov_b32 %[nr], -1\n\t"
                 "Lr%=:\n\t"
                 "s_andn2_b32 %[tmp], %[sch], %[u]\n\t"
                 "s_cmp_eq_u32 %[tmp], %[nr]\n\ts_cbranch_scc1 Ld%=\n\t"
                 "s_mov_b32 %[nr], %[tmp]\n\ts_mov_b32 %[u], %[ns]\n\t"
                 "s_cmp_eq_u32 %[tmp], 0\n\ts_cbranch_scc1 Lx%=\n\t"
                 "Lh%=:\n\t"
                 "s_ff1_i32_b32 %[t], %[tmp]\n\t"
                 "s_bitset0_b32 %[tmp], %[t]\n\t"
                 "v_readlane_b32 %[t], %[ve], %[t]\n\t"
                 "s_min_u32 %[u], %[u], %[t]\n\t"
                 "s_cmp_lg_u32 %[tmp], 0\n\ts_cbranch_scc1 Lh%=\n\t"
                 "Lx%=:\n\t"
                 "s_cmp_le_u32 %[u], %[xav]\n\ts_cbranch_scc1 Ld%=\n\t"
                 SC_TR
                 "v_mov_b32 v21, %[u]\n\t"
                 "s_mov_b32 exec_lo, 1\n\tds_write_b32 %[xva], v21\n\ts_mov_b32 exec_lo, -1\n\t"
                 "s_mov_b32 %[xav], %[u]\n\t"
                 "s_cmp_eq_u32 %[u], 32\n\ts_cbranch_scc1 Le%=\n\t"
                 "Ld%=:\n\t"
                 "s_sub_u32 %[t], s90, %[t0]\n\t"
                 "s_cmp_lt_u32 %[t], %[dl]\n\ts_cbranch_scc1 Ls%=\n\t"
                 "Lf%=:\n\t"
                 "s_or_b32 %[t], %[thr], 0x80\n\t"
                 "s_mov_b64 exec, -1\n\t"
                 "v_mov_b32 v21, %[t]\n\tv_mov_b32 v23, 1\n\t"
                 "buffer_store_dword v21, %[vf], %[rs], %[f0] offen sc0 sc1\n\t"
                 "ds_write_b32 %[xva], v23 offset:%[fo]\n\t"
                 "s_mov_b32 exec_hi, 0\n\t"
                 "s_mov_b32 %[arr], 0xfefefefe\n\t"
                 "s_branch Lt%=\n\t"
                 "Le%=:\n\t"
                 "s_mov_b64 exec, -1\n\t"
                 "s_waitcnt vmcnt(0) lgkmcnt(0)"
                 : [sch] "=&s"(sch), [arr] "=&s"(arr), [ann] "=&s"(ann), [tmp] "=&s"(tmp), [ta] "=&s"(ta), [tb] "=&s"(tb),
                   [xav] "=&s"(xav), [ns] "=&s"(ns), [nr] "=&s"(nr), [t] "=&s"(t), [u] "=&s"(u), [thr] "=&s"(thr),
                   [vt] "+v"(vt), [ve] "+v"(ve), [vlast] "+v"(vlast) SC_OUT
                 : [rs] "s"(rs), [cto] "s"(WS_CTR + b * 128 + 4), [fp] "s"(WS_FLAGS + mblk * 2048 + (b & 7) * 256),
                   [vid] "v"(vid), [vi] "v"(vi), [per] "s"(7 * SLOT4 / 4 + SLACK), [tab] "s"(lds0 + LDS_CTRL + 4 * C_TAB),
                   [mg] "s"(MG), [la] "s"(LA), [xva] "v"(lds0 + LDS_CTRL + 4 * C_XAV), [t0] "s"((unsigned)ts0),
                   [dl] "s"((unsigned)DEADLINE_TICKS), [f0] "s"(WS_FLAGS), [vf] "v"(vl * 16), [fo] "n"(4 * (C_FAIL - C_XAV))
                 : "memory", "scc", "vcc", "s72", "s73", "s74", "s75", "s76", "s77", "s78", "s79", "s80", "s81", "s82",
                   "s83", "s84", "s85", "s86", "s87", "s90", "s91", "v20", "v21", "v22", "v23", "v24");
    return;
  }

#define LANDED(r) \
    "s_mov_b32 m0, %[fa]\n\ts_mov_b64 exec, 1\n\tbuffer_load_dword off, " r ", %[oob] lds\n\t" \
    "s_mov_b64 exec, -1\n\ts_add_u32 %[fa], %[fa], 16\n\t"
#define GAP(r) "buffer_load_dword %[vg], off, " r ", %[oob]\n\t"
#define NGAP 2
#define GAP2(r) GAP(r)
#define GAP4(r)
#define LSPIN
#define LNAP
#define LPRIO 2

  if (wv == 1) {
    const unsigned cto = WS_CTR + b * 128 + 4;
    unsigned c0;
    __builtin_amdgcn_s_setprio(LPRIO);
    const v4i rx = make_rsrc(ka[0], ka[1], 2u << 20);
    const unsigned vx1 = vy - 1024, vx2 = vx + 2 * (4096 - 1024), vx3 = vx1 + 2 * (4096 - 1024);
    unsigned fa = lds0 + LDS_CTRL + 4 * (C_FLG + 2), vg;
    GATE();
    {
      unsigned m = lds0 + LDS_X, so = (rank * 512 + mblk * 64) * 512 + r0 * 128, so2;
      asm volatile("s_buffer_load_dword %[c0], %[rs], %[cto] glc\n\t"
                   "Lxl%=:\n\t"
                   "s_mov_b32 m0, %[m]\n\ts_add_u32 %[so2], %[so], 16384\n\t"
                   "buffer_load_dwordx4 %[v0], %[rx], %[so] offen lds\n\t"
                   "buffer_load_dwordx4 %[v1], %[rx], %[so] offen offset:1024 lds\n\t"
                   "buffer_load_dwordx4 %[v2], %[rx], %[so] offen offset:2048 lds\n\t"
                   "buffer_load_dwordx4 %[v3], %[rx], %[so] offen offset:3072 lds\n\t"
                   "s_add_u32 m0, m0, 4096\n\ts_add_u32 %[so], %[so], 128\n\t"
                   "buffer_load_dwordx4 %[v0], %[rx], %[so2] offen lds\n\t"
                   "buffer_load_dwordx4 %[v1], %[rx], %[so2] offen offset:1024 lds\n\t"
                   "buffer_load_dwordx4 %[v2], %[rx], %[so2] offen offset:2048 lds\n\t"
                   "buffer_load_dwordx4 %[v3], %[rx], %[so2] offen offset:3072 lds\n\t"
                   LANDED("%[rx]") GAP("%[rx]")
                   "s_add_u32 %[m], %[m], 8192\n\ts_bitset0_b32 %[so], 9\n\t"
                   "s_cmp_lg_u32 %[m], %[me]\n\ts_cbranch_scc1 Lxl%=\n\t"
                   "s_waitcnt lgkmcnt(0)"
                   : [c0] "=&s"(c0), [m] "+s"(m), [so] "+s"(so), [so2] "=&s"(so2), [fa] "+s"(fa), [vg] "=&v"(vg)
                   : [v0] "v"(vx), [v1] "v"(vx1), [v2] "v"(vx2), [v3] "v"(vx3), [rx] "s"(rx), [rs] "s"(rs), [cto] "s"(cto),
                     [me] "s"(lds0 + LDS_X + 32768u), [oob] "s"(OOB)
                   : "memory", "scc");
    }
    const unsigned xb = (c0 & 3) * WS_STAGE + mblk * 262144 - 8192;
    const unsigned mb = lds0 + LDS_X;
    const v4i rsx = rs;
    const unsigned t0 = now32();
    unsigned ni = 4, dn = 0, rc = 0, want = 0x01010101u, spin = 0, ta = lds0 + LDS_CTRL + 4 * (C_TAB + 4);
    unsigned kx = 0x80808080u + (NXS - 1 - 4) * 0x01010101u, mm = mb + (4 % NXS) * 8192;
    for (;;) {
      unsigned so, so2, t, u, av, vt;
      v4u vc;
      asm volatile("s_branch Lxq%=\n\tLxp%=:\n\t" LNAP "Lxq%=:\n\t"
                   "ds_read_b128 %[vc], %[ca]\n\tds_read_b32 %[vt], %[ta]\n\t"
                   "Lxw%=:\n\t"
                   "s_mov_b32 m0, %[mm]\n\t"
                   LSPIN
                   "s_waitcnt lgkmcnt(0)\n\t"
                   "v_readfirstlane_b32 %[av], v22\n\tv_readfirstlane_b32 %[dn], v20\n\tv_readfirstlane_b32 %[u], v21\n\t"
                   "v_readfirstlane_b32 %[so], %[vt]\n\t"
                   "s_cmp_eq_u32 %[dn], 0x20202020\n\ts_cbranch_scc1 Lout%=\n\t"
                   "s_cmp_eq_u32 %[u], %[want]\n\ts_cbranch_scc1 Lout%=\n\t"
                   "s_cmp_ge_u32 %[ni], %[av]\n\ts_cbranch_scc1 Lxp%=\n\t"
                   "s_add_u32 %[t], %[dn], %[kx]\n\ts_andn2_b32 %[t], 0x80808080, %[t]\n\ts_cbranch_scc1 Lxp%=\n\t"
                   "s_lshl_b32 %[so], %[so], 13\n\ts_add_u32 %[so], %[so], %[xb]\n\t"
                   "v_add_u32 %[ta], 4, %[ta]\n\t"
                   "buffer_load_dwordx4 %[vl], %[rs], %[so] offen " XAUX " lds\n\t"
                   "buffer_load_dwordx4 %[vl], %[rs], %[so] offen offset:1024 " XAUX " lds\n\t"
                   GAP4("%[rs]")
                   "buffer_load_dwordx4 %[vl], %[rs], %[so] offen offset:2048 " XAUX " lds\n\t"
                   "buffer_load_dwordx4 %[vl], %[rs], %[so] offen offset:3072 " XAUX " lds\n\t"
                   "s_add_u32 m0, m0, 4096\n\ts_add_u32 %[so2], %[so], 4096\n\t"
                   GAP2("%[rs]")
                   "buffer_load_dwordx4 %[vl], %[rs], %[so2] offen " XAUX " lds\n\t"
                   "buffer_load_dwordx4 %[vl], %[rs], %[so2] offen offset:1024 " XAUX " lds\n\t"
                   GAP4("%[rs]")
                   "buffer_load_dwordx4 %[vl], %[rs], %[so2] offen offset:2048 " XAUX " lds\n\t"
                   "buffer_load_dwordx4 %[vl], %[rs], %[so2] offen offset:3072 " XAUX " lds\n\t"
                   "ds_read_b128 %[vc], %[ca]\n\tds_read_b32 %[vt], %[ta]\n\t"
                   LANDED("%[rs]") GAP("%[rs]")
                   "s_add_u32 %[ni], %[ni], 1\n\t"
                   "s_sub_u32 %[kx], %[kx], 0x01010101\n\t"
                   "s_add_u32 %[mm], %[mm], 8192\n\ts_cmp_eq_u32 %[mm], %[me]\n\ts_cselect_b32 %[mm], %[mb], %[mm]\n\t"
                   "s_branch Lxw%=\n\t"
                   "Lout%=:\n\t"
                   "s_waitcnt vmcnt(0)"
                   : [ni] "+s"(ni), [vg] "+v"(vg), [dn] "+s"(dn), [av] "=&s"(av), [spin] "+s"(spin), [kx] "+s"(kx),
                     [mm] "+s"(mm), [fa] "+s"(fa), [so] "=&s"(so), [so2] "=&s"(so2), [t] "=&s"(t), [u] "=&s"(u),
                     [vc] "=&{v[20:23]}"(vc), [vt] "=&v"(vt), [ta] "+v"(ta)
                   : [vl] "v"(vl), [rs] "s"(rsx), [want] "s"(want), [mb] "s"(mb), [me] "s"(mb + NXS * 8192u), [xb] "s"(xb),
                     [ca] "v"(lds0 + LDS_CTRL), [oob] "s"(OOB)
                   : "memory", "scc");
      if (dn == 0x20202020u || u != want) break;
      if (now32() - t0 > (unsigned)DEADLINE_TICKS) {
        asm volatile("buffer_store_dword %0, %3, %1, %2 offen sc0 sc1\n\ts_waitcnt vmcnt(0)"
                     :: "v"((((c0 >> 2) + 1) << 11) | 0x80u), "s"(rs), "s"(WS_FLAGS), "v"(vl * 16) : "memory");
        lds_st1(lds0 + LDS_CTRL + 4 * C_FAIL, 1u);
      } else {
        const unsigned e = dn & 31u, m = mb + (e % NXS) * 8192;
        const unsigned s0 = xb + 8192 * __builtin_amdgcn_readfirstlane(
                                            *(volatile unsigned LDS_AS*)(smem + LDS_CTRL + 4 * (C_TAB + e)));
        asm volatile("s_mov_b32 m0, %0\n\ts_nop 4\n\t"
                     "buffer_load_dwordx4 %1, %2, %3 offen sc0 sc1 lds\n\t"
                     "buffer_load_dwordx4 %1, %2, %3 offen offset:1024 sc0 sc1 lds\n\t"
                     "buffer_load_dwordx4 %1, %2, %3 offen offset:2048 sc0 sc1 lds\n\t"
                     "buffer_load_dwordx4 %1, %2, %3 offen offset:3072 sc0 sc1 lds\n\t"
                     "s_add_u32 m0, m0, 4096\n\ts_nop 0\n\t"
                     "buffer_load_dwordx4 %1, %2, %4 offen sc0 sc1 lds\n\t"
                     "buffer_load_dwordx4 %1, %2, %4 offen offset:1024 sc0 sc1 lds\n\t"
                     "buffer_load_dwordx4 %1, %2, %4 offen offset:2048 sc0 sc1 lds\n\t"
                     "buffer_load_dwordx4 %1, %2, %4 offen offset:3072 sc0 sc1 lds\n\t"
                     "s_waitcnt vmcnt(0)"
                     :: "s"(m), "v"(vl), "s"(rs), "s"(s0), "s"(s0 + 4096u) : "memory", "scc");
      }
      ++rc;
      want = ((rc + 1) & 255u) * 0x01010101u;
      lds_st1(lds0 + LDS_CTRL + 4 * C_XFIX, rc);
    }
    asm volatile("s_mov_b64 exec, 1\n\tbuffer_store_dword %0, off, %1, %2 sc1\n\ts_mov_b64 exec, -1"
                 :: "v"(next_ctr(c0)), "s"(rs), "s"(cto) : "memory");
    STAMP(3);
    STAMPV(30, (1ull << 62) + rc);
    asm volatile("s_waitcnt vmcnt(0)" ::: "memory");
    return;
  }

  if (wv < 4) {
    const unsigned h = wv - 2;
    __builtin_amdgcn_s_setprio(LPRIO);
    const v4i rw = make_rsrc(ka[2], ka[3], 16u << 20);
    const unsigned sw0 = ((lane & 7) ^ ((lane >> 4) & 3)) * 16;
    const unsigned v0 = (lane >> 3) * 4096 + sw0 + (sw0 & 64 ? hx : 0);
    const unsigned v1 = v0 + 8 * 4096 - 1024;
    const unsigned v2 = (lane >> 3) * 4096 + (sw0 ^ 64) + (sw0 & 64 ? 0 : hx) + 16 * 4096 - 2048;
    const unsigned v3 = v2 + 8 * 4096 - 1024;
    const unsigned mb = lds0 + h * 8192;
    unsigned ti, dn, kx, so, so2, t, mm, vg, vd, vt, spin = 0, loc = rank * 4 + 1 + r0, ta = lds0 + LDS_CTRL + 4 * C_TAB;
    unsigned fa = lds0 + LDS_CTRL + 4 * (C_FLG + h);
    GATE();
    asm volatile("s_mov_b32 %[ti], 0\n\ts_mov_b32 %[mm], %[mb]\n\ts_mov_b32 %[kx], %[k0]\n\t"
                 "s_mov_b32 m0, %[mb]\n\ts_mov_b32 %[so], %[loc]\n\t"
                 "s_branch Lwi%=\n\t"
                 "Lwp%=:\n\t" LNAP
                 "ds_read_b32 %[vd], %[ca]\n\tds_read_b32 %[vt], %[ta]\n\t"
                 "Lww%=:\n\t"
                 "s_mov_b32 m0, %[mm]\n\t"
                 LSPIN
                 "s_cmp_lt_u32 %[ti], 4\n\t"
                 "s_waitcnt lgkmcnt(0)\n\t"
                 "v_readfirstlane_b32 %[so], %[vt]\n\tv_readfirstlane_b32 %[dn], %[vd]\n\t"
                 "s_cselect_b32 %[so], %[loc], %[so]\n\t"
                 "s_cmp_eq_u32 %[so], 0\n\ts_cbranch_scc1 Lwp%=\n\t"
                 "s_add_u32 %[t], %[dn], %[kx]\n\ts_andn2_b32 %[t], 0x80808080, %[t]\n\ts_cbranch_scc1 Lwp%=\n\t"
                 "Lwi%=:\n\t"
                 "s_lshl_b32 %[so], %[so], 19\n\ts_add_u32 %[so], %[so], %[wb]\n\t"
                 "v_add_u32 %[ta], 4, %[ta]\n\t"
                 "buffer_load_dwordx4 %[v0], %[rw], %[so] offen lds\n\t"
                 "buffer_load_dwordx4 %[v1], %[rw], %[so] offen offset:1024 lds\n\t"
                 GAP4("%[rw]")
                 "buffer_load_dwordx4 %[v2], %[rw], %[so] offen offset:2048 lds\n\t"
                 "buffer_load_dwordx4 %[v3], %[rw], %[so] offen offset:3072 lds\n\t"
                 "s_add_u32 m0, m0, 4096\n\ts_add_u32 %[so2], %[so], 131072\n\t"
                 GAP2("%[rw]")
                 "buffer_load_dwordx4 %[v0], %[rw], %[so2] offen lds\n\t"
                 "buffer_load_dwordx4 %[v1], %[rw], %[so2] offen offset:1024 lds\n\t"
                 GAP4("%[rw]")
                 "buffer_load_dwordx4 %[v2], %[rw], %[so2] offen offset:2048 lds\n\t"
                 "buffer_load_dwordx4 %[v3], %[rw], %[so2] offen offset:3072 lds\n\t"
                 "ds_read_b32 %[vd], %[ca]\n\tds_read_b32 %[vt], %[ta]\n\t"
                 LANDED("%[rw]") GAP("%[rw]")
                 "s_add_u32 %[ti], %[ti], 1\n\ts_add_u32 %[loc], %[loc], 1\n\t"
                 "s_cmp_eq_u32 %[loc], %[le]\n\ts_cselect_b32 %[loc], %[lb], %[loc]\n\t"
                 "s_sub_u32 %[kx], %[kx], 0x01010101\n\t"
                 "s_add_u32 %[mm], %[mm], 16384\n\ts_cmp_eq_u32 %[mm], %[me]\n\ts_cselect_b32 %[mm], %[mb], %[mm]\n\t"
                 "s_cmp_lg_u32 %[ti], 32\n\ts_cbranch_scc1 Lww%=\n\t"
                 "Lout%=:\n\t"
                 "s_waitcnt vmcnt(0) lgkmcnt(0)"
                 : [ti] "=&s"(ti), [mm] "=&s"(mm), [fa] "+s"(fa), [dn] "=&s"(dn), [kx] "=&s"(kx), [so] "=&s"(so), [so2] "=&s"(so2),
                   [t] "=&s"(t), [vg] "=&v"(vg), [vd] "=&v"(vd), [vt] "=&v"(vt), [spin] "+s"(spin), [loc] "+s"(loc),
                   [ta] "+v"(ta)
                 : [v0] "v"(v0), [v1] "v"(v1), [v2] "v"(v2), [v3] "v"(v3), [rw] "s"(rw),
                   [mb] "s"(mb), [me] "s"(mb + NW * 16384u), [k0] "s"(0x80808080u + (NW - 1) * 0x01010101u),
                   [wb] "s"(h * 64 * 4096 + cb - (1u << 19)), [lb] "s"(rank * 4 + 1), [le] "s"(rank * 4 + 5),
                   [ca] "v"(lds0 + LDS_CTRL), [oob] "s"(OOB)
                 : "memory", "scc");
    STAMP(4 + h);
    return;
  }

  {
    const int cw = wv - 4;
    int sxi, swi;
    asm volatile("s_load_dword %0, %2, 0x0\n\ts_load_dword %1, %3, 0x0\n\ts_waitcnt lgkmcnt(0)"
                 : "=&s"(sxi), "=&s"(swi) : "s"(mk64(ka[4], ka[5])), "s"(mk64(ka[6], ka[7])) : "memory");
    float scale = __int_as_float(sxi) * __int_as_float(swi);
    const unsigned g = lane >> 4, qq = (lane & 15) >> 1, pp = lane & 1;
    const unsigned F = ((qq >> 1) & 3) | ((g & 1) << 2);
    const unsigned wa0 = lds0 + 128 * (16 * g + qq) + 8 * pp + 16 * ((2 * cw) ^ F);
    const unsigned xa0 = lds0 + LDS_X + (lane & 15) * 128 + 16 * (g ^ qq);
    const unsigned ca = lds0 + LDS_CTRL + 4 * C_FAIL, fl = lds0 + LDS_CTRL + 4 * C_FLG, da = lds0 + LDS_CTRL + 4 * C_DONE + cw;
    const __amdgpu_buffer_rsrc_t ro =
        __builtin_amdgcn_make_buffer_rsrc((void*)mk64(ka[8], ka[9]), 0, 512 * 4096 * 4, 0x00020000);
    unsigned o = ((mblk * 64 + (lane & 15)) * 4096 + cb + cw * 32 + (cw & 2 ? hx : 0) + g * 4) * 4;
    v4f acc[4][2];
    unsigned need, rdy, woff, xoff, spin, failv, thr, rc;
#define CPRIO 1
    __builtin_amdgcn_s_setprio(CPRIO);
#define POLL_NAP
#define SPIN_INIT
#define SPIN_STEP
    asm volatile(
        "s_mov_b32 %[need], 1\n\t"
        "s_mov_b32 %[woff], 0\n\t"
        "s_mov_b32 %[xoff], 0\n\t"
        "s_mov_b32 %[rc], 0\n\t"
        SPIN_INIT
        "v_mov_b32 v0, 0\n\t"
        "v_mov_b32 v1, 0\n\t"
        "v_mov_b32 v2, 0\n\t"
        "v_mov_b32 v3, 0\n\t"
        "v_mov_b32 v4, 0\n\t"
        "v_mov_b32 v5, 0\n\t"
        "v_mov_b32 v6, 0\n\t"
        "v_mov_b32 v7, 0\n\t"
        "v_mov_b32 v8, 0\n\t"
        "v_mov_b32 v9, 0\n\t"
        "v_mov_b32 v10, 0\n\t"
        "v_mov_b32 v11, 0\n\t"
        "v_mov_b32 v12, 0\n\t"
        "v_mov_b32 v13, 0\n\t"
        "v_mov_b32 v14, 0\n\t"
        "v_mov_b32 v15, 0\n\t"
        "v_mov_b32 v16, 0\n\t"
        "v_mov_b32 v17, 0\n\t"
        "v_mov_b32 v18, 0\n\t"
        "v_mov_b32 v19, 0\n\t"
        "v_mov_b32 v20, 0\n\t"
        "v_mov_b32 v21, 0\n\t"
        "v_mov_b32 v22, 0\n\t"
        "v_mov_b32 v23, 0\n\t"
        "v_mov_b32 v24, 0\n\t"
        "v_mov_b32 v25, 0\n\t"
        "v_mov_b32 v26, 0\n\t"
        "v_mov_b32 v27, 0\n\t"
        "v_mov_b32 v28, 0\n\t"
        "v_mov_b32 v29, 0\n\t"
        "v_mov_b32 v30, 0\n\t"
        "v_mov_b32 v31, 0\n\t"
        "v_mov_b32 v132, %[wa]\n\t"
        "v_xor_b32 v133, 16, %[wa]\n\t"
        "v_mov_b32 v134, %[xa]\n\t"
        "v_xor_b32 v144, 64, %[xa]\n\t"
        "v_mov_b32 v141, %[fl]\n\t"
        "s_branch LpollB%=\n\t"
        "LstepA%=:\n\t"
        "s_cmp_lg_u32 %[rdy], 0\n\t"
        "s_cbranch_scc1 LslowA%=\n\t"
        "v_mfma_f32_16x16x128_f8f6f4 v[0:3], v[32:39], v[48:55], v[0:3] cbsz:1\n\t"
        "ds_read_b128 v[96:99], v137\n\t"
        "ds_read_b128 v[100:103], v145\n\t"
        "s_add_u32 %[woff], %[woff], 16384\n\t"
        "s_cmp_eq_u32 %[woff], %[wend]\n\t"
        "s_cselect_b32 %[woff], 0, %[woff]\n\t"
        "v_mfma_f32_16x16x128_f8f6f4 v[8:11], v[32:39], v[56:63], v[8:11] cbsz:1\n\t"
        "ds_read_b128 v[104:107], v137 offset:2048\n\t"
        "ds_read_b128 v[108:111], v145 offset:2048\n\t"
        "s_add_u32 %[xoff], %[xoff], 8192\n\t"
        "s_cmp_eq_u32 %[xoff], %[xend]\n\t"
        "s_cselect_b32 %[xoff], 0, %[xoff]\n\t"
        "v_mfma_f32_16x16x128_f8f6f4 v[16:19], v[32:39], v[64:71], v[16:19] cbsz:1\n\t"
        "ds_read_b128 v[112:115], v137 offset:4096\n\t"
        "ds_read_b128 v[116:119], v145 offset:4096\n\t"
        "v_add_u32 v132, %[woff], %[wa]\n\t"
        "v_add_u32 v134, %[xoff], %[xa]\n\t"
        "v_xor_b32 v133, 16, v132\n\t"
        "v_xor_b32 v144, 64, v134\n\t"
        "v_mfma_f32_16x16x128_f8f6f4 v[24:27], v[32:39], v[72:79], v[24:27] cbsz:1\n\t"
        "ds_read_b128 v[120:123], v137 offset:6144\n\t"
        "ds_read_b128 v[124:127], v145 offset:6144\n\t"
        "v_mov_b32 v138, %[need]\n\t"
        "s_cmp_gt_u32 %[need], 4\n\t"
        "s_cselect_b32 %[thr], 1, 0\n\t"
        "s_waitcnt lgkmcnt(8)\n\t"
        "v_mfma_f32_16x16x128_f8f6f4 v[4:7], v[40:47], v[48:55], v[4:7] cbsz:1\n\t"
        "ds_read_b64_tr_b8 v[80:81], v135\n\t"
        "ds_read_b64_tr_b8 v[82:83], v135 offset:1024\n\t"
        "ds_read_b64_tr_b8 v[84:85], v135 offset:8192\n\t"
        "s_add_u32 %[need], %[need], 1\n\t"
        "v_add_u32 v141, 16, v141\n\t"
        "v_mfma_f32_16x16x128_f8f6f4 v[12:15], v[40:47], v[56:63], v[12:15] cbsz:1\n\t"
        "ds_read_b64_tr_b8 v[86:87], v135 offset:9216\n\t"
        "ds_read_b128 v[128:131], v141\n\t"
        "ds_read_b64_tr_b8 v[88:89], v136\n\t"
        "v_mfma_f32_16x16x128_f8f6f4 v[20:23], v[40:47], v[64:71], v[20:23] cbsz:1\n\t"
        "s_waitcnt lgkmcnt(10)\n\t"
        "ds_read_b64_tr_b8 v[90:91], v136 offset:1024\n\t"
        "ds_read_b64_tr_b8 v[92:93], v136 offset:8192\n\t"
        "ds_read_b64_tr_b8 v[94:95], v136 offset:9216\n\t"
        "v_min3_u32 v139, v96, v100, v104\n\t"
        "v_mfma_f32_16x16x128_f8f6f4 v[28:31], v[40:47], v[72:79], v[28:31] cbsz:1\n\t"
        "LvalA%=:\n\t"
        "s_waitcnt lgkmcnt(9)\n\t"
        "v_min3_u32 v139, v139, v108, v112\n\t"
        "v_min3_u32 v140, v116, v120, v124\n\t"
        "v_min_u32 v139, v139, v140\n\t"
        "v_cmp_gt_u32 vcc, %[thr], v139\n\t"
        "s_cbranch_vccnz LbadA%=\n\t"
        "LtailA%=:\n\t"
        "s_mov_b64 exec, 1\n\t"
        "ds_write_b8 %[da], v138\n\t"
        "s_mov_b64 exec, -1\n\t"
        "s_waitcnt lgkmcnt(5)\n\t"
        "v_or3_b32 v128, v128, v129, v130\n\t"
        "s_nop 0\n\t"
        "v_readfirstlane_b32 %[rdy], v128\n\t"
        "LstepB%=:\n\t"
        "s_cmp_lg_u32 %[rdy], 0\n\t"
        "s_cbranch_scc1 LslowB%=\n\t"
        "v_mfma_f32_16x16x128_f8f6f4 v[0:3], v[80:87], v[96:103], v[0:3] cbsz:1\n\t"
        "ds_read_b128 v[48:51], v134\n\t"
        "ds_read_b128 v[52:55], v144\n\t"
        "s_add_u32 %[woff], %[woff], 16384\n\t"
        "s_cmp_eq_u32 %[woff], %[wend]\n\t"
        "s_cselect_b32 %[woff], 0, %[woff]\n\t"
        "v_mfma_f32_16x16x128_f8f6f4 v[8:11], v[80:87], v[104:111], v[8:11] cbsz:1\n\t"
        "ds_read_b128 v[56:59], v134 offset:2048\n\t"
        "ds_read_b128 v[60:63], v144 offset:2048\n\t"
        "s_add_u32 %[xoff], %[xoff], 8192\n\t"
        "s_cmp_eq_u32 %[xoff], %[xend]\n\t"
        "s_cselect_b32 %[xoff], 0, %[xoff]\n\t"
        "v_mfma_f32_16x16x128_f8f6f4 v[16:19], v[80:87], v[112:119], v[16:19] cbsz:1\n\t"
        "ds_read_b128 v[64:67], v134 offset:4096\n\t"
        "ds_read_b128 v[68:71], v144 offset:4096\n\t"
        "v_add_u32 v135, %[woff], %[wa]\n\t"
        "v_add_u32 v137, %[xoff], %[xa]\n\t"
        "v_xor_b32 v136, 16, v135\n\t"
        "v_xor_b32 v145, 64, v137\n\t"
        "v_mfma_f32_16x16x128_f8f6f4 v[24:27], v[80:87], v[120:127], v[24:27] cbsz:1\n\t"
        "ds_read_b128 v[72:75], v134 offset:6144\n\t"
        "ds_read_b128 v[76:79], v144 offset:6144\n\t"
        "v_mov_b32 v138, %[need]\n\t"
        "s_cmp_gt_u32 %[need], 4\n\t"
        "s_cselect_b32 %[thr], 1, 0\n\t"
        "s_waitcnt lgkmcnt(8)\n\t"
        "v_mfma_f32_16x16x128_f8f6f4 v[4:7], v[88:95], v[96:103], v[4:7] cbsz:1\n\t"
        "ds_read_b64_tr_b8 v[32:33], v132\n\t"
        "ds_read_b64_tr_b8 v[34:35], v132 offset:1024\n\t"
        "ds_read_b64_tr_b8 v[36:37], v132 offset:8192\n\t"
        "s_add_u32 %[need], %[need], 1\n\t"
        "v_add_u32 v141, 16, v141\n\t"
        "v_mfma_f32_16x16x128_f8f6f4 v[12:15], v[88:95], v[104:111], v[12:15] cbsz:1\n\t"
        "ds_read_b64_tr_b8 v[38:39], v132 offset:9216\n\t"
        "ds_read_b128 v[128:131], v141\n\t"
        "ds_read_b64_tr_b8 v[40:41], v133\n\t"
        "v_mfma_f32_16x16x128_f8f6f4 v[20:23], v[88:95], v[112:119], v[20:23] cbsz:1\n\t"
        "s_waitcnt lgkmcnt(10)\n\t"
        "ds_read_b64_tr_b8 v[42:43], v133 offset:1024\n\t"
        "ds_read_b64_tr_b8 v[44:45], v133 offset:8192\n\t"
        "ds_read_b64_tr_b8 v[46:47], v133 offset:9216\n\t"
        "v_min3_u32 v139, v48, v52, v56\n\t"
        "v_mfma_f32_16x16x128_f8f6f4 v[28:31], v[88:95], v[120:127], v[28:31] cbsz:1\n\t"
        "LvalB%=:\n\t"
        "s_waitcnt lgkmcnt(9)\n\t"
        "v_min3_u32 v139, v139, v60, v64\n\t"
        "v_min3_u32 v140, v68, v72, v76\n\t"
        "v_min_u32 v139, v139, v140\n\t"
        "v_cmp_gt_u32 vcc, %[thr], v139\n\t"
        "s_cbranch_vccnz LbadB%=\n\t"
        "LtailB%=:\n\t"
        "s_mov_b64 exec, 1\n\t"
        "ds_write_b8 %[da], v138\n\t"
        "s_mov_b64 exec, -1\n\t"
        "s_waitcnt lgkmcnt(5)\n\t"
        "v_or3_b32 v128, v128, v129, v130\n\t"
        "s_nop 0\n\t"
        "v_readfirstlane_b32 %[rdy], v128\n\t"
        "s_branch LstepA%=\n\t"
        "LslowA%=:\n\t"
        "v_mfma_f32_16x16x128_f8f6f4 v[0:3], v[32:39], v[48:55], v[0:3] cbsz:1\n\t"
        "v_mfma_f32_16x16x128_f8f6f4 v[8:11], v[32:39], v[56:63], v[8:11] cbsz:1\n\t"
        "v_mfma_f32_16x16x128_f8f6f4 v[16:19], v[32:39], v[64:71], v[16:19] cbsz:1\n\t"
        "v_mfma_f32_16x16x128_f8f6f4 v[24:27], v[32:39], v[72:79], v[24:27] cbsz:1\n\t"
        "s_waitcnt lgkmcnt(1)\n\t"
        "v_mfma_f32_16x16x128_f8f6f4 v[4:7], v[40:47], v[48:55], v[4:7] cbsz:1\n\t"
        "v_mfma_f32_16x16x128_f8f6f4 v[12:15], v[40:47], v[56:63], v[12:15] cbsz:1\n\t"
        "v_mfma_f32_16x16x128_f8f6f4 v[20:23], v[40:47], v[64:71], v[20:23] cbsz:1\n\t"
        "v_mfma_f32_16x16x128_f8f6f4 v[28:31], v[40:47], v[72:79], v[28:31] cbsz:1\n\t"
        "LpollA%=:\n\t"
        "ds_read_b128 v[128:131], v141\n\t"
        SPIN_STEP
        "s_waitcnt lgkmcnt(0)\n\t"
        "v_or3_b32 v128, v128, v129, v130\n\t"
        "s_nop 0\n\t"
        "v_readfirstlane_b32 %[rdy], v128\n\t"
        "s_cmp_eq_u32 %[rdy], 0\n\t"
        "s_cbranch_scc1 LgoA%=\n\t"
        POLL_NAP
        "s_branch LpollA%=\n\t"
        "LgoA%=:\n\t"
        "ds_read_b128 v[96:99], v137\n\t"
        "ds_read_b128 v[100:103], v145\n\t"
        "ds_read_b128 v[104:107], v137 offset:2048\n\t"
        "ds_read_b128 v[108:111], v145 offset:2048\n\t"
        "ds_read_b128 v[112:115], v137 offset:4096\n\t"
        "ds_read_b128 v[116:119], v145 offset:4096\n\t"
        "ds_read_b128 v[120:123], v137 offset:6144\n\t"
        "ds_read_b128 v[124:127], v145 offset:6144\n\t"
        "v_mov_b32 v138, %[need]\n\t"
        "s_cmp_gt_u32 %[need], 4\n\t"
        "s_cselect_b32 %[thr], 1, 0\n\t"
        "s_add_u32 %[need], %[need], 1\n\t"
        "v_add_u32 v141, 16, v141\n\t"
        "ds_read_b64_tr_b8 v[80:81], v135\n\t"
        "ds_read_b64_tr_b8 v[82:83], v135 offset:1024\n\t"
        "ds_read_b64_tr_b8 v[84:85], v135 offset:8192\n\t"
        "ds_read_b64_tr_b8 v[86:87], v135 offset:9216\n\t"
        "ds_read_b128 v[128:131], v141\n\t"
        "ds_read_b64_tr_b8 v[88:89], v136\n\t"
        "s_add_u32 %[woff], %[woff], 16384\n\t"
        "s_cmp_eq_u32 %[woff], %[wend]\n\t"
        "s_cselect_b32 %[woff], 0, %[woff]\n\t"
        "s_add_u32 %[xoff], %[xoff], 8192\n\t"
        "s_cmp_eq_u32 %[xoff], %[xend]\n\t"
        "s_cselect_b32 %[xoff], 0, %[xoff]\n\t"
        "v_add_u32 v132, %[woff], %[wa]\n\t"
        "v_add_u32 v134, %[xoff], %[xa]\n\t"
        "v_xor_b32 v133, 16, v132\n\t"
        "v_xor_b32 v144, 64, v134\n\t"
        "s_waitcnt lgkmcnt(10)\n\t"
        "ds_read_b64_tr_b8 v[90:91], v136 offset:1024\n\t"
        "ds_read_b64_tr_b8 v[92:93], v136 offset:8192\n\t"
        "ds_read_b64_tr_b8 v[94:95], v136 offset:9216\n\t"
        "v_min3_u32 v139, v96, v100, v104\n\t"
        "s_branch LvalA%=\n\t"
        "LslowB%=:\n\t"
        "v_mfma_f32_16x16x128_f8f6f4 v[0:3], v[80:87], v[96:103], v[0:3] cbsz:1\n\t"
        "v_mfma_f32_16x16x128_f8f6f4 v[8:11], v[80:87], v[104:111], v[8:11] cbsz:1\n\t"
        "v_mfma_f32_16x16x128_f8f6f4 v[16:19], v[80:87], v[112:119], v[16:19] cbsz:1\n\t"
        "v_mfma_f32_16x16x128_f8f6f4 v[24:27], v[80:87], v[120:127], v[24:27] cbsz:1\n\t"
        "s_waitcnt lgkmcnt(1)\n\t"
        "v_mfma_f32_16x16x128_f8f6f4 v[4:7], v[88:95], v[96:103], v[4:7] cbsz:1\n\t"
        "v_mfma_f32_16x16x128_f8f6f4 v[12:15], v[88:95], v[104:111], v[12:15] cbsz:1\n\t"
        "v_mfma_f32_16x16x128_f8f6f4 v[20:23], v[88:95], v[112:119], v[20:23] cbsz:1\n\t"
        "s_cmp_eq_u32 %[need], 33\n\t"
        "v_mfma_f32_16x16x128_f8f6f4 v[28:31], v[88:95], v[120:127], v[28:31] cbsz:1\n\t"
        "s_cbranch_scc1 Lend%=\n\t"
        "LpollB%=:\n\t"
        "ds_read_b128 v[128:131], v141\n\t"
        SPIN_STEP
        "s_waitcnt lgkmcnt(0)\n\t"
        "v_or3_b32 v128, v128, v129, v130\n\t"
        "s_nop 0\n\t"
        "v_readfirstlane_b32 %[rdy], v128\n\t"
        "s_cmp_eq_u32 %[rdy], 0\n\t"
        "s_cbranch_scc1 LgoB%=\n\t"
        POLL_NAP
        "s_branch LpollB%=\n\t"
        "LgoB%=:\n\t"
        "ds_read_b128 v[48:51], v134\n\t"
        "ds_read_b128 v[52:55], v144\n\t"
        "ds_read_b128 v[56:59], v134 offset:2048\n\t"
        "ds_read_b128 v[60:63], v144 offset:2048\n\t"
        "ds_read_b128 v[64:67], v134 offset:4096\n\t"
        "ds_read_b128 v[68:71], v144 offset:4096\n\t"
        "ds_read_b128 v[72:75], v134 offset:6144\n\t"
        "ds_read_b128 v[76:79], v144 offset:6144\n\t"
        "v_mov_b32 v138, %[need]\n\t"
        "s_cmp_gt_u32 %[need], 4\n\t"
        "s_cselect_b32 %[thr], 1, 0\n\t"
        "s_add_u32 %[need], %[need], 1\n\t"
        "v_add_u32 v141, 16, v141\n\t"
        "ds_read_b64_tr_b8 v[32:33], v132\n\t"
        "ds_read_b64_tr_b8 v[34:35], v132 offset:1024\n\t"
        "ds_read_b64_tr_b8 v[36:37], v132 offset:8192\n\t"
        "ds_read_b64_tr_b8 v[38:39], v132 offset:9216\n\t"
        "ds_read_b128 v[128:131], v141\n\t"
        "ds_read_b64_tr_b8 v[40:41], v133\n\t"
        "s_add_u32 %[woff], %[woff], 16384\n\t"
        "s_cmp_eq_u32 %[woff], %[wend]\n\t"
        "s_cselect_b32 %[woff], 0, %[woff]\n\t"
        "s_add_u32 %[xoff], %[xoff], 8192\n\t"
        "s_cmp_eq_u32 %[xoff], %[xend]\n\t"
        "s_cselect_b32 %[xoff], 0, %[xoff]\n\t"
        "v_add_u32 v135, %[woff], %[wa]\n\t"
        "v_add_u32 v137, %[xoff], %[xa]\n\t"
        "v_xor_b32 v136, 16, v135\n\t"
        "v_xor_b32 v145, 64, v137\n\t"
        "s_waitcnt lgkmcnt(10)\n\t"
        "ds_read_b64_tr_b8 v[42:43], v133 offset:1024\n\t"
        "ds_read_b64_tr_b8 v[44:45], v133 offset:8192\n\t"
        "ds_read_b64_tr_b8 v[46:47], v133 offset:9216\n\t"
        "v_min3_u32 v139, v48, v52, v56\n\t"
        "s_branch LvalB%=\n\t"
        "LbadA%=:\n\t"
        "s_add_u32 %[rc], %[rc], 1\n\t"
        "v_mov_b32 v139, %[rc]\n\t"
        "s_mov_b64 exec, 1\n\t"
        "ds_write_b8 %[da], v139 offset:4\n\t"
        "s_mov_b64 exec, -1\n\t"
        "LbpA%=:\n\t"
        "ds_read2_b32 v[142:143], %[ca] offset0:0 offset1:1\n\t"
        SPIN_STEP
        "s_waitcnt lgkmcnt(0)\n\t"
        "v_readfirstlane_b32 vcc_lo, v142\n\t"
        "v_readfirstlane_b32 vcc_hi, v143\n\t"
        "s_cmp_eq_u32 vcc_lo, 0\n\t"
        "s_cbranch_scc1 LbqA%=\n\t"
        "v_mov_b32 v131, 1\n\t"
        "s_branch LtailA%=\n\t"
        "LbqA%=:\n\t"
        "s_cmp_lg_u32 vcc_hi, %[rc]\n\t"
        "s_cbranch_scc1 LbpA%=\n\t"
        "ds_read_b128 v[96:99], v137\n\t"
        "ds_read_b128 v[100:103], v145\n\t"
        "ds_read_b128 v[104:107], v137 offset:2048\n\t"
        "ds_read_b128 v[108:111], v145 offset:2048\n\t"
        "ds_read_b128 v[112:115], v137 offset:4096\n\t"
        "ds_read_b128 v[116:119], v145 offset:4096\n\t"
        "ds_read_b128 v[120:123], v137 offset:6144\n\t"
        "ds_read_b128 v[124:127], v145 offset:6144\n\t"
        "s_waitcnt lgkmcnt(0)\n\t"
        "v_min3_u32 v139, v96, v100, v104\n\t"
        "s_branch LvalA%=\n\t"
        "LbadB%=:\n\t"
        "s_add_u32 %[rc], %[rc], 1\n\t"
        "v_mov_b32 v139, %[rc]\n\t"
        "s_mov_b64 exec, 1\n\t"
        "ds_write_b8 %[da], v139 offset:4\n\t"
        "s_mov_b64 exec, -1\n\t"
        "LbpB%=:\n\t"
        "ds_read2_b32 v[142:143], %[ca] offset0:0 offset1:1\n\t"
        SPIN_STEP
        "s_waitcnt lgkmcnt(0)\n\t"
        "v_readfirstlane_b32 vcc_lo, v142\n\t"
        "v_readfirstlane_b32 vcc_hi, v143\n\t"
        "s_cmp_eq_u32 vcc_lo, 0\n\t"
        "s_cbranch_scc1 LbqB%=\n\t"
        "v_mov_b32 v131, 1\n\t"
        "s_branch LtailB%=\n\t"
        "LbqB%=:\n\t"
        "s_cmp_lg_u32 vcc_hi, %[rc]\n\t"
        "s_cbranch_scc1 LbpB%=\n\t"
        "ds_read_b128 v[48:51], v134\n\t"
        "ds_read_b128 v[52:55], v144\n\t"
        "ds_read_b128 v[56:59], v134 offset:2048\n\t"
        "ds_read_b128 v[60:63], v144 offset:2048\n\t"
        "ds_read_b128 v[64:67], v134 offset:4096\n\t"
        "ds_read_b128 v[68:71], v144 offset:4096\n\t"
        "ds_read_b128 v[72:75], v134 offset:6144\n\t"
        "ds_read_b128 v[76:79], v144 offset:6144\n\t"
        "s_waitcnt lgkmcnt(0)\n\t"
        "v_min3_u32 v139, v48, v52, v56\n\t"
        "s_branch LvalB%=\n\t"
        "Lend%=:\n\t"
        "s_nop 15\n\t"
        : "=&{v[0:3]}"(acc[0][0]), "=&{v[4:7]}"(acc[0][1]), "=&{v[8:11]}"(acc[1][0]), "=&{v[12:15]}"(acc[1][1]),
          "=&{v[16:19]}"(acc[2][0]), "=&{v[20:23]}"(acc[2][1]), "=&{v[24:27]}"(acc[3][0]), "=&{v[28:31]}"(acc[3][1]),
          "=&{v131}"(failv), [sc] "+v"(scale), [o] "+v"(o),
          [need] "=&s"(need), [rdy] "=&s"(rdy), [woff] "=&s"(woff), [xoff] "=&s"(xoff), [spin] "=&s"(spin),
          [thr] "=&s"(thr), [rc] "=&s"(rc)
        : [wa] "v"(wa0), [xa] "v"(xa0), [ca] "v"(ca), [da] "v"(da), [fl] "v"(fl), [wend] "s"(NW * 16384u), [xend] "s"(NXS * 8192u)
        : "memory", "scc", "vcc",
          "v32", "v33", "v34", "v35", "v36", "v37", "v38", "v39", "v40", "v41", "v42", "v43", "v44", "v45", "v46", "v47",
          "v48", "v49", "v50", "v51", "v52", "v53", "v54", "v55", "v56", "v57", "v58", "v59", "v60", "v61", "v62", "v63",
          "v64", "v65", "v66", "v67", "v68", "v69", "v70", "v71", "v72", "v73", "v74", "v75", "v76", "v77", "v78", "v79",
          "v80", "v81", "v82", "v83", "v84", "v85", "v86", "v87", "v88", "v89", "v90", "v91", "v92", "v93", "v94", "v95",
          "v96", "v97", "v98", "v99", "v100", "v101", "v102", "v103", "v104", "v105", "v106", "v107", "v108", "v109",
          "v110", "v111", "v112", "v113", "v114", "v115", "v116", "v117", "v118", "v119", "v120", "v121", "v122", "v123",
          "v124", "v125", "v126", "v127", "v128", "v129", "v130", "v132", "v133", "v134", "v135", "v136", "v137", "v138",
          "v139", "v140", "v141", "v142", "v143", "v144", "v145", "v146", "v147");
    TS(ts6);
    scale = failv ? __builtin_nanf("") : scale;
    const float nl2e = scale * -1.4426950408889634f, clip = 60.0f * 1.4426950408889634f;
#pragma unroll
    for (int mt = 0; mt < 4; ++mt) {
#pragma unroll
      for (int c2 = 0; c2 < 2; ++c2) {
        v4f y = acc[mt][c2] * scale;
        v4f r = acc[mt][c2] * nl2e;
#pragma unroll
        for (int e = 0; e < 4; ++e) r[e] = __builtin_amdgcn_exp2f(__builtin_amdgcn_fmed3f(r[e], -clip, clip));
        r = r + 1.0f;
#pragma unroll
        for (int e = 0; e < 4; ++e) r[e] = __builtin_amdgcn_rcpf(r[e]);
        r = r * y;
        __builtin_amdgcn_raw_buffer_store_b128(__builtin_bit_cast(v4u, r), ro, o + mt * 262144 + c2 * 64, 0, OAUX);
      }
    }
  }
}

}

extern "C" void kernel_launch(int rank, int nranks, ncclComm_t comm, hipStream_t stream,
                              void* const* d_in, const int* in_sizes, int n_in,
                              void* d_out, int out_size,
                              const int* in_dtypes, int out_dtype,
                              void* const* d_out_all,
                              void* const* d_ws_all, size_t ws_bytes) {
  (void)comm; (void)in_sizes; (void)in_dtypes; (void)out_dtype; (void)d_out_all;
  if (nranks != NRANKS || (unsigned)rank >= (unsigned)NRANKS || n_in < 4 || ws_bytes < WS_NEEDED ||
      out_size != 512 * 4096) {
    fprintf(stderr, "a2a_gemm: unsupported configuration\n");
    return;
  }
  HIP_CHECK(hipFuncSetAttribute((const void*)a2a_gemm_kernel, hipFuncAttributeMaxDynamicSharedMemorySize, LDS_BYTES));
  Args a;
  a.x = (const unsigned char*)d_in[0];
  a.w = (const unsigned char*)d_in[1];
  a.sx = (const float*)d_in[2];
  a.sw = (const float*)d_in[3];
  a.out = (float*)d_out;
  a.ws_self = (unsigned char*)d_ws_all[rank];
  a.rank = rank;
  a.nranks = nranks;
  a.pad[0] = a.pad[1] = 0;
  for (int p = 0; p < NRANKS; ++p) {
    a.ws[p] = (unsigned char*)d_ws_all[p];
    a.wn[p] = (unsigned char*)d_ws_all[(rank + 1 + p) % NRANKS];
  }
  hipLaunchKernelGGL(a2a_gemm_kernel, dim3(NWG), dim3(NTHREADS), LDS_BYTES, stream, a);
  HIP_CHECK(hipGetLastError());
}
```
